```python
import numpy as np

kernel_name = "gpu_a2a_gemm_m2048_k2048_n2048_bf16_silu"


ABSMAX_RTOL = 0.008

COMPUTE_DTYPE = "bfloat16"
EPILOGUE = "silu"
IN_QUANT = None
OUT_QUANT = None
IN_DTYPES = {"x": "bfloat16", "w_mat": "bfloat16"}
WORLD_SIZE = 8

def _to_bf16(x):
    u = np.ascontiguousarray(x, dtype=np.float32).view(np.uint32)
    return ((u + 0x7FFF + ((u >> 16) & 1)) & 0xFFFF0000).view(np.float32)

def setup_inputs():

    x = _to_bf16(np.random.randn(2048, 2048).astype(np.float32))
    w_mat = _to_bf16((np.random.randn(2048, 2048) / np.sqrt(2048)).astype(np.float32))
    return {"x": x, "w_mat": w_mat}

def _epilogue(y):
    return (y / (1.0 + np.exp(-np.clip(y, -60.0, 60.0)))).astype(np.float32)

def reference(x, w_mat):
    y = (x @ w_mat).astype(np.float32)
    return _epilogue(y)
```

```cpp
#include <hip/hip_runtime.h>
#include <rccl/rccl.h>
#include <cstdio>

#define HIP_CHECK(expr) do { hipError_t _e = (expr); if (_e != hipSuccess) { \
    fprintf(stderr, "HIP error %s at %s:%d\n", hipGetErrorString(_e), __FILE__, __LINE__); \
    return; } } while (0)

typedef __attribute__((ext_vector_type(8))) short s16x8;
typedef __attribute__((ext_vector_type(4))) short s16x4;
typedef __attribute__((ext_vector_type(4))) float f32x4;
typedef __attribute__((ext_vector_type(4))) unsigned u32x4;
typedef __attribute__((ext_vector_type(2))) unsigned u32x2;

static constexpr int NR = 8;
static constexpr int KSH = 256;
static constexpr int NN = 2048;
static constexpr size_t SLOT = 128 * 1024;
static constexpr size_t OFF_FLAG = 2 * NR * SLOT;
static constexpr size_t OFF_CNT = OFF_FLAG + NR * 32 * 128;
static constexpr size_t OFF_TMO = OFF_CNT + 256 * 128;
static constexpr unsigned long long DEADLINE_TICKS = 50000000ull;

__device__ __forceinline__ int swz(int r) { return ((r >> 1) & 1) | (((r >> 3) & 1) << 1); }

__device__ __forceinline__ unsigned short f2bf(float f) {
    unsigned u = __float_as_uint(f);
    return (unsigned short)((u + 0x7FFFu + ((u >> 16) & 1u)) >> 16);
}

#define STAMP(i) do {} while (0)
__global__ __launch_bounds__(512, 2)
void a2a_gemm_silu(int rank, const unsigned short* __restrict__ x, const unsigned short* __restrict__ w,
                   unsigned short* __restrict__ out, void* const* __restrict__ ws_all) {
    __shared__ __attribute__((aligned(16))) char lds[8 * 16384];
    const int tid = threadIdx.x, lane = tid & 63;
    const int wv = __builtin_amdgcn_readfirstlane(tid >> 6);
    const int b = blockIdx.x;
    char* myws = (char*)ws_all[rank];
    __shared__ unsigned s_call;
    STAMP(0);
    if (tid == 0) {
        unsigned* cnt = (unsigned*)(myws + OFF_CNT + (size_t)b * 128);
        s_call = __hip_atomic_fetch_add(cnt, 1u, __ATOMIC_RELAXED, __HIP_MEMORY_SCOPE_AGENT) + 1u;
    }
    const bool pusher = b < (NR - 1) * 32;
    const unsigned p = (unsigned)(rank + 1 + (b >> 5)) % NR;
    const int j = b & 31;
    u32x4 v = {0u, 0u, 0u, 0u};
    if (pusher && tid < 256) v = *(const u32x4*)((const char*)x + ((size_t)p * 256 + j * 8) * 512 + tid * 16);

    const int xcd = b & 7, idx = b >> 3;
    const int ntile = xcd * 4 + (idx & 3), mtile = idx >> 2;
    const int n0 = ntile * 64, m0 = mtile * 32;
    char* myl = lds + wv * 16384;
    s16x8 bf[8][4];

    __syncthreads();
    const unsigned call = s_call;
    const size_t par = (size_t)(call & 1u) * (NR * SLOT);
    if (pusher && tid < 256) {
        char* dst = (char*)ws_all[p] + par + (size_t)rank * SLOT + j * 4096 + tid * 16;
        asm volatile("global_store_dwordx4 %0, %1, off sc0 sc1" :: "v"(dst), "v"(v) : "memory");
    }
#pragma unroll
    for (int h = 0; h < 2; ++h) {
        const int kb = wv * KSH + h * 128;
#pragma unroll
        for (int i = 0; i < 16; ++i) {
            const int r = 8 * i + (lane >> 3);
            const int pos = lane & 7;
            const int c = (pos >> 1) ^ swz(r);
            const unsigned short* src = w + (size_t)(kb + r) * NN + n0 + c * 16 + (pos & 1) * 8;
            __builtin_amdgcn_global_load_lds((const __attribute__((address_space(1))) void*)src,
                                             (__attribute__((address_space(3))) void*)(myl + i * 1024), 16, 0, 0);
        }
        if (h == 0 && pusher) {
            asm volatile("s_waitcnt vmcnt(16)" ::: "memory");
            __builtin_amdgcn_s_barrier();
            if (tid == 0)
                __hip_atomic_store((unsigned*)((char*)ws_all[p] + OFF_FLAG + (size_t)(rank * 32 + j) * 128), call,
                                   __ATOMIC_RELAXED, __HIP_MEMORY_SCOPE_SYSTEM);
            STAMP(1);
        }
        asm volatile("s_waitcnt vmcnt(0)" ::: "memory");
        const int g = lane >> 4, q = (lane & 15) >> 2, pp = lane & 3;
#pragma unroll
        for (int ks = 0; ks < 4; ++ks)
#pragma unroll
            for (int nb = 0; nb < 4; ++nb) {
                s16x4 t0, t1;
                {
                    const int r = 32 * ks + 8 * g + q;
                    t0 = __builtin_amdgcn_ds_read_tr16_b64_v4i16(
                        (__attribute__((address_space(3))) s16x4*)(myl + r * 128 + ((nb ^ swz(r)) * 32) + pp * 8));
                }
                {
                    const int r = 32 * ks + 8 * g + 4 + q;
                    t1 = __builtin_amdgcn_ds_read_tr16_b64_v4i16(
                        (__attribute__((address_space(3))) s16x4*)(myl + r * 128 + ((nb ^ swz(r)) * 32) + pp * 8));
                }
                bf[h * 4 + ks][nb] = __builtin_shufflevector(t0, t1, 0, 1, 2, 3, 4, 5, 6, 7);
            }
        asm volatile("s_waitcnt lgkmcnt(0)" ::: "memory");
    }

    STAMP(2);
    const char* abase;
    bool ok = true;
    if (wv == rank) {
        abase = (const char*)x + (size_t)rank * SLOT;
    } else {
        abase = myws + par + (size_t)wv * SLOT;
        unsigned* tmo = (unsigned*)(myws + OFF_TMO);
        const unsigned* flag = (const unsigned*)(myws + OFF_FLAG + (size_t)(wv * 32 + mtile * 4 + (lane & 3)) * 128);
        unsigned long long t0 = __builtin_amdgcn_s_memrealtime();
        while (true) {
            unsigned f = __hip_atomic_load(flag, __ATOMIC_RELAXED, __HIP_MEMORY_SCOPE_SYSTEM);
            if (__all(f >= call)) break;
            if (__hip_atomic_load(tmo, __ATOMIC_RELAXED, __HIP_MEMORY_SCOPE_AGENT) == call ||
                __builtin_amdgcn_s_memrealtime() - t0 > DEADLINE_TICKS) {
                __hip_atomic_store(tmo, call, __ATOMIC_RELAXED, __HIP_MEMORY_SCOPE_AGENT);
                ok = false;
                break;
            }
            __builtin_amdgcn_s_sleep(1);
        }
        asm volatile("" ::: "memory");
    }

    STAMP(3);
    f32x4 acc[2][4];
#pragma unroll
    for (int i = 0; i < 2; ++i)
#pragma unroll
        for (int j = 0; j < 4; ++j) acc[i][j] = (f32x4){0.f, 0.f, 0.f, 0.f};
    if (ok) {
#pragma unroll
        for (int mb = 0; mb < 2; ++mb) {
            s16x8 af[8];
            const char* ap = abase + (size_t)(m0 + mb * 16 + (lane & 15)) * 512 + (lane >> 4) * 16;
#pragma unroll
            for (int ks = 0; ks < 8; ++ks)
                asm volatile("global_load_dwordx4 %0, %1, off" : "=v"(af[ks]) : "v"(ap + ks * 64) : "memory");
            asm volatile("s_waitcnt vmcnt(0)" ::: "memory");
            __builtin_amdgcn_sched_barrier(0);
#pragma unroll
            for (int ks = 0; ks < 8; ++ks)
#pragma unroll
                for (int nb = 0; nb < 4; ++nb)
                    acc[mb][nb] = __builtin_amdgcn_mfma_f32_16x16x32_bf16(bf[ks][nb], af[ks], acc[mb][nb], 0, 0, 0);
        }
    }
    STAMP(4);
#pragma unroll
    for (int mb = 0; mb < 2; ++mb)
#pragma unroll
        for (int nb = 0; nb < 4; ++nb)
            *(f32x4*)(myl + (mb * 16 + (lane & 15)) * 272 + (nb * 16 + (lane >> 4) * 4) * 4) = acc[mb][nb];
    __syncthreads();
    {
        const int m = tid >> 4, n4 = (tid & 15) * 4;
        f32x4 s = (f32x4){0.f, 0.f, 0.f, 0.f};
#pragma unroll
        for (int v = 0; v < 8; ++v) s += *(const f32x4*)(lds + v * 16384 + m * 272 + n4 * 4);
        const bool bad = __hip_atomic_load((unsigned*)(myws + OFF_TMO), __ATOMIC_RELAXED, __HIP_MEMORY_SCOPE_AGENT) == call;
        unsigned short o[4];
#pragma unroll
        for (int j = 0; j < 4; ++j) {
            float y = s[j];
            float c = fminf(fmaxf(y, -60.f), 60.f);
            float r = y / (1.f + __expf(-c));
            o[j] = bad ? (unsigned short)0x7FC0 : f2bf(r);
        }
        u32x2 pk = {(unsigned)o[0] | ((unsigned)o[1] << 16), (unsigned)o[2] | ((unsigned)o[3] << 16)};
        *(u32x2*)(out + (size_t)(m0 + m) * NN + n0 + n4) = pk;
    }
    STAMP(5);
}

extern "C" void kernel_launch(int rank, int nranks, ncclComm_t comm,
                              hipStream_t stream,
                              void* const* d_in, const int* in_sizes, int n_in,
                              void* d_out, int out_size,
                              const int* in_dtypes, int out_dtype,
                              void* const* d_out_all,
                              void* const* d_ws_all, size_t ws_bytes) {
    if (nranks != NR || ws_bytes < OFF_TMO + 128) return;
    hipLaunchKernelGGL(a2a_gemm_silu, dim3(256), dim3(512), 0, stream, rank,
                       (const unsigned short*)d_in[0], (const unsigned short*)d_in[1],
                       (unsigned short*)d_out, d_ws_all);
    HIP_CHECK(hipGetLastError());
}
```
